```python
import numpy as np

kernel_name = "gpu_a2a_gemm_m8192_k8192_n8192_f32_relu"


ABSMAX_RTOL = 0.001

COMPUTE_DTYPE = "float32"
EPILOGUE = "relu"
IN_QUANT = None
OUT_QUANT = None
IN_DTYPES = {"x": "float32", "w_mat": "float32"}
WORLD_SIZE = 8

def setup_inputs():

    x = np.random.randn(8192, 8192).astype(np.float32)
    w_mat = (np.random.randn(8192, 8192) / np.sqrt(8192)).astype(np.float32)
    return {"x": x, "w_mat": w_mat}

def _epilogue(y):
    return np.maximum(y, 0.0).astype(np.float32)

def reference(x, w_mat):
    y = (x @ w_mat).astype(np.float32)
    return _epilogue(y)
```

```cpp
#include <hip/hip_runtime.h>
#include <rccl/rccl.h>
#include <cstdio>

#define HIP_CHECK(expr) do { hipError_t _e = (expr); if (_e != hipSuccess) { \
    fprintf(stderr, "HIP error %s at %s:%d\n", hipGetErrorString(_e), __FILE__, __LINE__); \
    return; } } while (0)

#define LAS __attribute__((address_space(3)))
#define GAS __attribute__((address_space(1)))
typedef _Float16 h8 __attribute__((ext_vector_type(8)));
typedef float f16v __attribute__((ext_vector_type(16)));
typedef float f4 __attribute__((ext_vector_type(4)));

namespace {

constexpr int NRANKS = 8;
constexpr int MLOC = 1024, NN = 8192, KK = 8192, KC = 1024;
constexpr int BM = 256, BN = 128, BK = 64;
constexpr int T_TILES = KK / BK;
constexpr int NUNITS = 64;
constexpr int BSTAGES = 4;
constexpr int A_TILE_BYTES = BM * BK * 2;
constexpr int B_BYTES = BN * BK * 2;
constexpr int CTL_OFF = BSTAGES * B_BYTES;
constexpr int LDS_BYTES = CTL_OFF + 64;
constexpr int PDEPTH = 3;
constexpr int NWG = 256;

constexpr size_t WS_TICKET = 0, WS_TMO = 128, WS_FLAGS = 4096;
constexpr size_t WS_STAGE = 1u << 20;
constexpr size_t SLOT_BYTES = (size_t)MLOC * KK * 2;
constexpr size_t WS_NEED = WS_STAGE + 2 * SLOT_BYTES;

constexpr long long DEADLINE = 40000000;

__device__ __forceinline__ unsigned long long rt() { return __builtin_amdgcn_s_memrealtime(); }
__device__ __forceinline__ unsigned lds_ld(LAS unsigned* p) { return __hip_atomic_load(p, __ATOMIC_RELAXED, __HIP_MEMORY_SCOPE_WORKGROUP); }
__device__ __forceinline__ void lds_st(LAS unsigned* p, unsigned v) { __hip_atomic_store(p, v, __ATOMIC_RELAXED, __HIP_MEMORY_SCOPE_WORKGROUP); }
__device__ __forceinline__ int fsw(int r) { return ((r >> 2) & 7) ^ ((r >> 1) & 1); }
__device__ __forceinline__ int ktile_of(int i, int rank) {
    if (i < 16) return rank * 16 + i;
    const int j = i - 16, pair = j >> 1, h = j & 1, s = pair / 7, q = pair - s * 7 + 1;
    return (((rank + q) & 7) << 4) + 2 * s + h;
}
#define SB() __builtin_amdgcn_sched_barrier(0)
#define LGKM0() __builtin_amdgcn_s_waitcnt(0xC07F)

__global__ __launch_bounds__(512, 2) void a2a_gemm_relu(const float* __restrict__ x, const float* __restrict__ w, float* __restrict__ out,
                                                        unsigned char* const* __restrict__ ws_all, int rank)
{
    extern __shared__ __attribute__((aligned(16))) unsigned char lds[];
    LAS unsigned char* L8 = (LAS unsigned char*)lds;
    LAS unsigned* CTL = (LAS unsigned*)(L8 + CTL_OFF);
    asm volatile("" :: "s"(x), "s"(w), "s"(out), "s"(ws_all), "s"(rank));
    const int tid = threadIdx.x, wave = __builtin_amdgcn_readfirstlane(tid >> 6), lane = tid & 63;
    const int b = blockIdx.x, xcd = b & 7, idx = b >> 3, mt = idx & 3, nt = xcd * 8 + (idx >> 2);
    GAS unsigned char* const ws = (GAS unsigned char*)ws_all[rank];
    if (tid < 16) CTL[tid] = 0u;
    __syncthreads();
    if (tid == 0) {
        const unsigned ticket = __hip_atomic_fetch_add((GAS unsigned*)(ws + WS_TICKET), 1u, __ATOMIC_RELAXED, __HIP_MEMORY_SCOPE_AGENT);
        CTL[8] = ticket / NWG + 1;
    }
    __syncthreads();
    const unsigned call = __builtin_amdgcn_readfirstlane(CTL[8]);
    GAS unsigned* const tmo = (GAS unsigned*)(ws + WS_TMO);

    if (wave < 4) {
        const int l31 = lane & 31, hi = lane >> 5;
        const int f = fsw(l31);
        unsigned boff[4];
#pragma unroll
        for (int kk = 0; kk < 4; ++kk) boff[kk] = l31 * 128 + (((2 * kk + hi) ^ f) << 4);
        f16v acc[2][4];
#pragma unroll
        for (int i = 0; i < 2; ++i)
#pragma unroll
            for (int j = 0; j < 4; ++j)
#pragma unroll
                for (int r = 0; r < 16; ++r) acc[i][j][r] = 0.f;
        h8 fa[8][2];
        h8 fb[2][4];
        unsigned soff = 0;
        int dead = 0;
        const unsigned voff = lane * 16;
        const GAS unsigned char* abase = ws + WS_STAGE + (size_t)(call & 1) * SLOT_BYTES + (size_t)mt * T_TILES * A_TILE_BYTES + wave * 8192;
#define ALOAD(slot, ptile, kk) do { \
            asm volatile("global_load_dwordx4 %0, %1, %2 offset:%3" : "=v"(fa[slot][0]) : "v"(voff), "s"(ptile), "n"((kk) * 1024) : "memory"); \
            asm volatile("global_load_dwordx4 %0, %1, %2 offset:%3" : "=v"(fa[slot][1]) : "v"(voff), "s"((ptile) + 4096), "n"((kk) * 1024) : "memory"); } while (0)
#define AWAIT(slot, n) asm volatile("s_waitcnt vmcnt(%2)" : "+v"(fa[slot][0]), "+v"(fa[slot][1]) : "n"(n) : "memory")
#define MM(slot, set, rb, cb) acc[rb][cb] = __builtin_amdgcn_mfma_f32_32x32x16_f16(fa[slot][rb], fb[set][cb], acc[rb][cb], 0, 0, 0)
#define BRD(set, so, kk, cb) fb[set][cb] = *(const LAS h8*)(L8 + (so) + boff[kk] + (cb) * 4096)
        auto slow_wait = [&](int word0, int nwords, unsigned need) {
            const unsigned long long t0 = rt(); LGKM0(); int sp = 0;
            for (;;) {
                bool ok = true;
                for (int i = 0; i < nwords; ++i) ok = ok && (__builtin_amdgcn_readfirstlane(lds_ld(CTL + word0 + i)) >= need);
                if (ok) break;
                if ((++sp & 255) == 0) {
                    if (lds_ld(CTL + 3) != 0u) { dead = 2; break; }
                    if ((long long)(rt() - t0) > DEADLINE) { dead = 1; lds_st(CTL + 3, 1u); break; }
                }
            }
            asm volatile("" ::: "memory");
        };
        auto atile = [&](int t) -> const GAS unsigned char* { return abase + (size_t)ktile_of(min(t, T_TILES - 1), rank) * A_TILE_BYTES; };
        if (__builtin_amdgcn_readfirstlane(lds_ld(CTL + 0)) < 2u) slow_wait(0, 1, 2u);
        { const GAS unsigned char* p0 = atile(0); const GAS unsigned char* p1 = atile(1);
          ALOAD(0, p0, 0); ALOAD(1, p0, 1); ALOAD(2, p0, 2); ALOAD(3, p0, 3); ALOAD(4, p1, 0); ALOAD(5, p1, 1); ALOAD(6, p1, 2); }
        if (__builtin_amdgcn_readfirstlane(lds_ld(CTL + 1)) < 1u || __builtin_amdgcn_readfirstlane(lds_ld(CTL + 2)) < 1u) slow_wait(1, 2, 1u);
#pragma unroll
        for (int cb = 0; cb < 4; ++cb) BRD(0, soff, 0, cb);
        unsigned rdy = lds_ld(CTL + 0);
        SB();
#define STEPI(slot, set, so_n, kk_n, aslot, aptr, akk) do { \
            MM(slot, set, 0, 0); MM(slot, set, 1, 0); BRD(1 - (set), so_n, kk_n, 0); SB(); \
            MM(slot, set, 0, 1); BRD(1 - (set), so_n, kk_n, 1); MM(slot, set, 1, 1); BRD(1 - (set), so_n, kk_n, 2); SB(); \
            MM(slot, set, 0, 2); BRD(1 - (set), so_n, kk_n, 3); MM(slot, set, 1, 2); SB(); \
            ALOAD(aslot, aptr, akk); SB(); \
            MM(slot, set, 0, 3); MM(slot, set, 1, 3); SB(); } while (0)
#define TILE(PAR, t) do { \
            const GAS unsigned char* pn = atile((t) + 1); const GAS unsigned char* pnn = atile((t) + 2); \
            { const unsigned needA = (unsigned)min((t) + 3, T_TILES); if (__builtin_amdgcn_readfirstlane(rdy) < needA) slow_wait(0, 1, needA); } \
            AWAIT((PAR) * 4 + 0, 12); STEPI((PAR) * 4 + 0, 0, soff, 1, (1 - (PAR)) * 4 + 3, pn, 3); \
            AWAIT((PAR) * 4 + 1, 12); STEPI((PAR) * 4 + 1, 1, soff, 2, (PAR) * 4 + 0, pnn, 0); \
            AWAIT((PAR) * 4 + 2, 12); STEPI((PAR) * 4 + 2, 0, soff, 3, (PAR) * 4 + 1, pnn, 1); \
            const unsigned pb0 = lds_ld(CTL + 1), pb1 = lds_ld(CTL + 2); rdy = lds_ld(CTL + 0); \
            AWAIT((PAR) * 4 + 3, 12); \
            asm volatile("" ::: "memory"); LGKM0(); \
            lds_st(CTL + 4 + wave, (unsigned)((t) + 1)); \
            unsigned nso = soff + B_BYTES; if (nso == BSTAGES * B_BYTES) nso = 0; \
            if ((t) + 1 < T_TILES && (__builtin_amdgcn_readfirstlane(pb0) < (unsigned)((t) + 2) || __builtin_amdgcn_readfirstlane(pb1) < (unsigned)((t) + 2))) slow_wait(1, 2, (unsigned)((t) + 2)); \
            asm volatile("" ::: "memory"); \
            STEPI((PAR) * 4 + 3, 1, nso, 0, (PAR) * 4 + 2, pnn, 2); \
            soff = nso; } while (0)
#pragma unroll 1
        for (int t = 0; t < T_TILES; t += 2) {
            TILE(0, t);
            TILE(1, t + 1);
            if (dead) break;
        }
        asm volatile("s_waitcnt vmcnt(0)" : "+v"(fa[0][0]), "+v"(fa[0][1]), "+v"(fa[1][0]), "+v"(fa[1][1]), "+v"(fa[2][0]), "+v"(fa[2][1]), "+v"(fa[3][0]), "+v"(fa[3][1]),
                     "+v"(fa[4][0]), "+v"(fa[4][1]), "+v"(fa[5][0]), "+v"(fa[5][1]), "+v"(fa[6][0]), "+v"(fa[6][1]), "+v"(fa[7][0]), "+v"(fa[7][1]) :: "memory");
        if (__hip_atomic_load(tmo, __ATOMIC_RELAXED, __HIP_MEMORY_SCOPE_AGENT) == call || lds_ld(CTL + 3) != 0u) dead = 2;
        if (dead == 1 && lane == 0) __hip_atomic_store(tmo, call, __ATOMIC_RELAXED, __HIP_MEMORY_SCOPE_AGENT);
        const float poison = dead ? __builtin_nanf("") : 0.f;
#pragma unroll
        for (int rb = 0; rb < 2; ++rb)
#pragma unroll
            for (int cb = 0; cb < 4; ++cb)
#pragma unroll
                for (int r = 0; r < 16; ++r) {
                    const int row = mt * BM + wave * 64 + rb * 32 + (r & 3) + 8 * (r >> 2) + 4 * hi;
                    const int col = nt * BN + cb * 32 + l31;
                    out[(size_t)row * NN + col] = fmaxf(acc[rb][cb][r], 0.f) + poison;
                }
    } else if (wave < 6) {
        const int h = wave - 4, g = lane & 31, hh = lane >> 5;
        f4 r[PDEPTH][16];
        unsigned woff[2][4];
#pragma unroll
        for (int q = 0; q < 2; ++q)
#pragma unroll
            for (int i = 0; i < 4; ++i) { const int n = 4 * g + i, c = 4 * h + 2 * q + hh; woff[q][i] = n * 128 + ((c ^ fsw(n)) << 4); }
        const unsigned voff = (unsigned)((4 * g + (size_t)(32 * h + 8 * hh) * NN) * 4);
        const float* wtile0 = w + (size_t)nt * BN;
        auto issue = [&](int slot, int u) {
            const float* p = wtile0 + (size_t)ktile_of(u, rank) * BK * NN;
#pragma unroll
            for (int q = 0; q < 2; ++q)
#pragma unroll
                for (int j = 0; j < 8; ++j) {
                    const float* sp = p + (size_t)(16 * q + j) * NN;
                    asm volatile("global_load_dwordx4 %0, %1, %2" : "=v"(r[slot][q * 8 + j]) : "v"(voff), "s"(sp) : "memory");
                }
        };
#define PWAIT(slot) asm volatile("s_waitcnt vmcnt(%16)" : "+v"(r[slot][0]), "+v"(r[slot][1]), "+v"(r[slot][2]), "+v"(r[slot][3]), "+v"(r[slot][4]), "+v"(r[slot][5]), "+v"(r[slot][6]), "+v"(r[slot][7]), \
                                     "+v"(r[slot][8]), "+v"(r[slot][9]), "+v"(r[slot][10]), "+v"(r[slot][11]), "+v"(r[slot][12]), "+v"(r[slot][13]), "+v"(r[slot][14]), "+v"(r[slot][15]) : "n"(NW) : "memory")
        int sidx = 0;
        auto wr = [&](int slot, int u) {
            const int need = u - BSTAGES + 1;
            auto isfree = [&]() -> bool { return (int)lds_ld(CTL + 4) >= need && (int)lds_ld(CTL + 5) >= need && (int)lds_ld(CTL + 6) >= need && (int)lds_ld(CTL + 7) >= need; };
            if (!isfree()) {
                int sp = 0; const unsigned long long t0 = rt(); LGKM0();
                while (!isfree()) {
                    __builtin_amdgcn_s_sleep(2);
                    if ((++sp & 255) == 0) { if (lds_ld(CTL + 3) != 0u) break; if ((long long)(rt() - t0) > DEADLINE) { lds_st(CTL + 3, 1u); break; } }
                }
            }
            asm volatile("" ::: "memory");
            const unsigned soff = sidx * B_BYTES;
#pragma unroll
            for (int q = 0; q < 2; ++q)
#pragma unroll
                for (int i = 0; i < 4; ++i) {
                    h8 v;
#pragma unroll
                    for (int j = 0; j < 8; ++j) v[j] = (_Float16)r[slot][q * 8 + j][i];
                    *(LAS h8*)(L8 + soff + woff[q][i]) = v;
                }
            asm volatile("s_waitcnt lgkmcnt(0)" ::: "memory");
            lds_st(CTL + 1 + h, (unsigned)(u + 1));
            if (++sidx == BSTAGES) sidx = 0;
        };
#pragma unroll
        for (int d = 0; d < PDEPTH; ++d) issue(d, d);
        constexpr int NFULL = (T_TILES / PDEPTH) * PDEPTH;
#pragma unroll 1
        for (int u0 = 0; u0 < NFULL; u0 += PDEPTH) {
#pragma unroll
            for (int d = 0; d < PDEPTH; ++d) {
                const int u = u0 + d;
                { constexpr int NW = 16 * (PDEPTH - 1); PWAIT(d); }
                wr(d, u);
                issue(d, min(u + PDEPTH, T_TILES - 1));
                SB();
            }
        }
#pragma unroll
        for (int d = 0; d < T_TILES - NFULL; ++d) { { constexpr int NW = 0; PWAIT(d); } wr(d, NFULL + d); }
        { constexpr int NW = 0; PWAIT(0); PWAIT(1); PWAIT(2); }
    } else if (wave == 6) {
        const int dest = b >> 5, rg = b & 31, mtd = rg >> 3, rb = rg & 7;
        GAS unsigned char* const dws = (GAS unsigned char*)ws_all[dest];
        GAS unsigned char* dst = dws + WS_STAGE + (size_t)(call & 1) * SLOT_BYTES + ((size_t)mtd * T_TILES + rank * 16) * A_TILE_BYTES + rb * 4096 + lane * 16;
        const float* src = x + (size_t)(32 * b + (lane & 31)) * KC + (lane >> 5) * 8;
        GAS unsigned* flag = (GAS unsigned*)(dws + WS_FLAGS + (size_t)((rank * 8) * 4 + mtd) * 128) + rb;
        f4 cur[16];
#pragma unroll
        for (int j = 0; j < 8; ++j) { cur[2 * j] = *(const f4*)(src + j * 16); cur[2 * j + 1] = *(const f4*)(src + j * 16 + 4); }
#pragma unroll 1
        for (int s = 0; s < 8; ++s) {
            h8 v[8];
#pragma unroll
            for (int j = 0; j < 8; ++j)
#pragma unroll
                for (int e = 0; e < 4; ++e) { v[j][e] = (_Float16)cur[2 * j][e]; v[j][4 + e] = (_Float16)cur[2 * j + 1][e]; }
            const float* nsrc = src + (size_t)min(s + 1, 7) * 128;
#pragma unroll
            for (int j = 0; j < 8; ++j) { cur[2 * j] = *(const f4*)(nsrc + j * 16); cur[2 * j + 1] = *(const f4*)(nsrc + j * 16 + 4); }
#pragma unroll
            for (int j = 0; j < 8; ++j)
                *(GAS h8*)(dst + (size_t)(2 * s + (j >> 2)) * A_TILE_BYTES + (j & 3) * 1024) = v[j];
            asm volatile("s_waitcnt vmcnt(0)" ::: "memory");
            __builtin_amdgcn_fence(__ATOMIC_RELEASE, "");
            asm volatile("s_waitcnt vmcnt(0)" ::: "memory");
            if (lane == 0) __hip_atomic_store(flag + s * 4 * 32, call, __ATOMIC_RELAXED, __HIP_MEMORY_SCOPE_SYSTEM);
        }
    } else {
        GAS unsigned* flags = (GAS unsigned*)(ws + WS_FLAGS);
        int un = 0;
        unsigned long long t0 = rt(); int sp = 0;
        while (un < NUNITS) {
            const int u = un + (lane >> 3);
            bool ok = true;
            if (u < NUNITS) {
                int p, s;
                if (u < 8) { p = rank; s = u; } else { const int v = u - 8; s = v / 7; p = (rank + (v - s * 7) + 1) & 7; }
                const unsigned fv = __hip_atomic_load(flags + (size_t)((p * 8 + s) * 4 + mt) * 32 + (lane & 7), __ATOMIC_RELAXED, __HIP_MEMORY_SCOPE_SYSTEM);
                ok = fv >= call;
            }
            const unsigned long long nr = __ballot(!ok);
            const int nready = nr ? (__builtin_ctzll(nr) >> 3) : 8;
            if (nready > 0) {
                __builtin_amdgcn_fence(__ATOMIC_ACQUIRE, "");
                asm volatile("s_waitcnt vmcnt(0)" ::: "memory");
                un = min(un + nready, NUNITS);
                lds_st(CTL + 0, (unsigned)(2 * un));
                t0 = rt(); sp = 0;
            } else {
                __builtin_amdgcn_s_sleep(2);
                if ((++sp & 63) == 0) {
                    if (lds_ld(CTL + 3) != 0u) break;
                    if (__hip_atomic_load(tmo, __ATOMIC_RELAXED, __HIP_MEMORY_SCOPE_AGENT) == call || (long long)(rt() - t0) > DEADLINE) {
                        __hip_atomic_store(tmo, call, __ATOMIC_RELAXED, __HIP_MEMORY_SCOPE_AGENT);
                        lds_st(CTL + 3, 1u);
                        break;
                    }
                }
            }
        }
    }
}

}

extern "C" void kernel_launch(int rank, int nranks, ncclComm_t comm, hipStream_t stream,
                              void* const* d_in, const int* in_sizes, int n_in,
                              void* d_out, int out_size,
                              const int* in_dtypes, int out_dtype,
                              void* const* d_out_all,
                              void* const* d_ws_all, size_t ws_bytes) {
    (void)comm; (void)in_sizes; (void)n_in; (void)out_size; (void)in_dtypes; (void)out_dtype; (void)d_out_all;
    if (nranks != NRANKS || (unsigned)rank >= (unsigned)NRANKS || ws_bytes < WS_NEED) {
        fprintf(stderr, "kernel_launch: unsupported configuration (nranks %d, rank %d, ws %zu)\n", nranks, rank, ws_bytes);
        return;
    }
    static bool attr_done[64] = {};
    int dev = 0;
    HIP_CHECK(hipGetDevice(&dev));
    if (!attr_done[dev & 63]) {
        HIP_CHECK(hipFuncSetAttribute((const void*)a2a_gemm_relu, hipFuncAttributeMaxDynamicSharedMemorySize, LDS_BYTES));
        attr_done[dev & 63] = true;
    }
    a2a_gemm_relu<<<NWG, 512, LDS_BYTES, stream>>>((const float*)d_in[0], (const float*)d_in[1], (float*)d_out,
                                                   (unsigned char* const*)d_ws_all, rank);
    HIP_CHECK(hipGetLastError());
}
```

```cpp
#include <hip/hip_runtime.h>
#include <rccl/rccl.h>
#include <cstdio>

#define HIP_CHECK(expr) do { hipError_t _e = (expr); if (_e != hipSuccess) { \
    fprintf(stderr, "HIP error %s at %s:%d\n", hipGetErrorString(_e), __FILE__, __LINE__); \
    return; } } while (0)

#define LAS __attribute__((address_space(3)))
#define GAS __attribute__((address_space(1)))
typedef _Float16 h8 __attribute__((ext_vector_type(8)));
typedef float f16v __attribute__((ext_vector_type(16)));
typedef float f4 __attribute__((ext_vector_type(4)));

namespace {

constexpr int NRANKS = 8;
constexpr int MLOC = 1024, NN = 8192, KK = 8192, KC = 1024;
constexpr int BM = 256, BN = 128, BK = 64;
constexpr int T_TILES = KK / BK;
constexpr int NUNITS = 64;
constexpr int BSTAGES = 4;
constexpr int A_TILE_BYTES = BM * BK * 2;
constexpr int B_BYTES = BN * BK * 2;
constexpr int CTL_OFF = BSTAGES * B_BYTES;
constexpr int LDS_BYTES = CTL_OFF + 64;
constexpr int PDEPTH = 3;
constexpr int NWG = 256;

constexpr size_t WS_TICKET = 0, WS_TMO = 128, WS_FLAGS = 4096;
constexpr size_t WS_STAGE = 1u << 20;
constexpr size_t SLOT_BYTES = (size_t)MLOC * KK * 2;
constexpr size_t WS_NEED = WS_STAGE + 2 * SLOT_BYTES;

constexpr long long DEADLINE = 40000000;

__device__ __forceinline__ unsigned long long rt() { return __builtin_amdgcn_s_memrealtime(); }
__device__ __forceinline__ unsigned lds_ld(LAS unsigned* p) { return __hip_atomic_load(p, __ATOMIC_RELAXED, __HIP_MEMORY_SCOPE_WORKGROUP); }
__device__ __forceinline__ void lds_st(LAS unsigned* p, unsigned v) { __hip_atomic_store(p, v, __ATOMIC_RELAXED, __HIP_MEMORY_SCOPE_WORKGROUP); }
__device__ __forceinline__ int fsw(int r) { return ((r >> 2) & 7) ^ ((r >> 1) & 1); }
__device__ __forceinline__ int ktile_of(int i, int rank) {
    if (i < 16) return rank * 16 + i;
    const int j = i - 16, pair = j >> 1, h = j & 1, s = pair / 7, q = pair - s * 7 + 1;
    return (((rank + q) & 7) << 4) + 2 * s + h;
}
#define SB() __builtin_amdgcn_sched_barrier(0)
#define LGKM0() __builtin_amdgcn_s_waitcnt(0xC07F)

__global__ __launch_bounds__(512, 2) void a2a_gemm_relu(const float* __restrict__ x, const float* __restrict__ w, float* __restrict__ out,
                                                        unsigned char* const* __restrict__ ws_all, int rank)
{
    extern __shared__ __attribute__((aligned(16))) unsigned char lds[];
    LAS unsigned char* L8 = (LAS unsigned char*)lds;
    LAS unsigned* CTL = (LAS unsigned*)(L8 + CTL_OFF);
    asm volatile("" :: "s"(x), "s"(w), "s"(out), "s"(ws_all), "s"(rank));
    const int tid = threadIdx.x, wave = __builtin_amdgcn_readfirstlane(tid >> 6), lane = tid & 63;
    const int b = blockIdx.x, xcd = b & 7, idx = b >> 3, mt = idx & 3, nt = xcd * 8 + (idx >> 2);
    GAS unsigned char* const ws = (GAS unsigned char*)ws_all[rank];
    if (tid < 16) CTL[tid] = 0u;
    __syncthreads();
    if (tid == 0) {
        const unsigned ticket = __hip_atomic_fetch_add((GAS unsigned*)(ws + WS_TICKET), 1u, __ATOMIC_RELAXED, __HIP_MEMORY_SCOPE_AGENT);
        CTL[8] = ticket / NWG + 1;
    }
    __syncthreads();
    const unsigned call = __builtin_amdgcn_readfirstlane(CTL[8]);
    GAS unsigned* const tmo = (GAS unsigned*)(ws + WS_TMO);

#define DBG(i, v)
    if (wave < 4) {
        const int l31 = lane & 31, hi = lane >> 5;
        const int f = fsw(l31);
        unsigned boff[4];
#pragma unroll
        for (int kk = 0; kk < 4; ++kk) boff[kk] = l31 * 128 + (((2 * kk + hi) ^ f) << 4);
        f16v acc[2][4];
#pragma unroll
        for (int i = 0; i < 2; ++i)
#pragma unroll
            for (int j = 0; j < 4; ++j)
#pragma unroll
                for (int r = 0; r < 16; ++r) acc[i][j][r] = 0.f;
        h8 fa[8][2];
        h8 fb[2][4];
        unsigned soff = 0;
        int dead = 0;
        const unsigned voff = lane * 16;
        const GAS unsigned char* abase = ws + WS_STAGE + (size_t)(call & 1) * SLOT_BYTES + (size_t)mt * T_TILES * A_TILE_BYTES + wave * 8192;
#define ALOAD(slot, ptile, kk) do { \
            asm volatile("global_load_dwordx4 %0, %1, %2 offset:%3" : "=v"(fa[slot][0]) : "v"(voff), "s"(ptile), "n"((kk) * 1024) : "memory"); \
            asm volatile("global_load_dwordx4 %0, %1, %2 offset:%3" : "=v"(fa[slot][1]) : "v"(voff), "s"((ptile) + 4096), "n"((kk) * 1024) : "memory"); } while (0)
#define AWAIT(slot, n) asm volatile("s_waitcnt vmcnt(%2)" : "+v"(fa[slot][0]), "+v"(fa[slot][1]) : "n"(n) : "memory")
#define MM(slot, set, rb, cb) acc[rb][cb] = __builtin_amdgcn_mfma_f32_32x32x16_f16(fa[slot][rb], fb[set][cb], acc[rb][cb], 0, 0, 0)
#define BRD(set, so, kk, cb) fb[set][cb] = *(const LAS h8*)(L8 + (so) + boff[kk] + (cb) * 4096)
        auto slow_wait = [&](int word0, int nwords, unsigned need) {
            const unsigned long long t0 = rt(); LGKM0(); int sp = 0;
            for (;;) {
                bool ok = true;
                for (int i = 0; i < nwords; ++i) ok = ok && (__builtin_amdgcn_readfirstlane(lds_ld(CTL + word0 + i)) >= need);
                if (ok) break;
                if ((++sp & 255) == 0) {
                    if (lds_ld(CTL + 3) != 0u) { dead = 2; break; }
                    if ((long long)(rt() - t0) > DEADLINE) { dead = 1; lds_st(CTL + 3, 1u); break; }
                }
            }
            asm volatile("" ::: "memory");
        };
        auto atile = [&](int t) -> const GAS unsigned char* { return abase + (size_t)ktile_of(min(t, T_TILES - 1), rank) * A_TILE_BYTES; };
        if (__builtin_amdgcn_readfirstlane(lds_ld(CTL + 0)) < 2u) slow_wait(0, 1, 2u);
        if (wave == 0) DBG(7, rt() - tstart);
        { const GAS unsigned char* p0 = atile(0); const GAS unsigned char* p1 = atile(1);
          ALOAD(0, p0, 0); ALOAD(1, p0, 1); ALOAD(2, p0, 2); ALOAD(3, p0, 3); ALOAD(4, p1, 0); ALOAD(5, p1, 1); ALOAD(6, p1, 2); }
        if (__builtin_amdgcn_readfirstlane(lds_ld(CTL + 1)) < 1u || __builtin_amdgcn_readfirstlane(lds_ld(CTL + 2)) < 1u) slow_wait(1, 2, 1u);
#pragma unroll
        for (int cb = 0; cb < 4; ++cb) BRD(0, soff, 0, cb);
        unsigned rdy = lds_ld(CTL + 0);
        SB();
#define STEPI(slot, set, so_n, kk_n, aslot, aptr, akk) do { \
            MM(slot, set, 0, 0); MM(slot, set, 1, 0); BRD(1 - (set), so_n, kk_n, 0); SB(); \
            MM(slot, set, 0, 1); BRD(1 - (set), so_n, kk_n, 1); MM(slot, set, 1, 1); BRD(1 - (set), so_n, kk_n, 2); SB(); \
            MM(slot, set, 0, 2); BRD(1 - (set), so_n, kk_n, 3); MM(slot, set, 1, 2); SB(); \
            ALOAD(aslot, aptr, akk); SB(); \
            MM(slot, set, 0, 3); MM(slot, set, 1, 3); SB(); } while (0)
#define TILE(PAR, t) do { \
            const GAS unsigned char* pn = atile((t) + 1); const GAS unsigned char* pnn = atile((t) + 2); \
            { const unsigned needA = (unsigned)min((t) + 3, T_TILES); if (__builtin_amdgcn_readfirstlane(rdy) < needA) slow_wait(0, 1, needA); } \
            AWAIT((PAR) * 4 + 0, 12); STEPI((PAR) * 4 + 0, 0, soff, 1, (1 - (PAR)) * 4 + 3, pn, 3); \
            AWAIT((PAR) * 4 + 1, 12); STEPI((PAR) * 4 + 1, 1, soff, 2, (PAR) * 4 + 0, pnn, 0); \
            AWAIT((PAR) * 4 + 2, 12); STEPI((PAR) * 4 + 2, 0, soff, 3, (PAR) * 4 + 1, pnn, 1); \
            const unsigned pb0 = lds_ld(CTL + 1), pb1 = lds_ld(CTL + 2); rdy = lds_ld(CTL + 0); \
            AWAIT((PAR) * 4 + 3, 12); \
            asm volatile("" ::: "memory"); LGKM0(); \
            lds_st(CTL + 4 + wave, (unsigned)((t) + 1)); \
            unsigned nso = soff + B_BYTES; if (nso == BSTAGES * B_BYTES) nso = 0; \
            if ((t) + 1 < T_TILES && (__builtin_amdgcn_readfirstlane(pb0) < (unsigned)((t) + 2) || __builtin_amdgcn_readfirstlane(pb1) < (unsigned)((t) + 2))) slow_wait(1, 2, (unsigned)((t) + 2)); \
            asm volatile("" ::: "memory"); \
            STEPI((PAR) * 4 + 3, 1, nso, 0, (PAR) * 4 + 2, pnn, 2); \
            soff = nso; } while (0)
#pragma unroll 1
        for (int t = 0; t < T_TILES; t += 2) {
            TILE(0, t);
            TILE(1, t + 1);
            if (dead) break;
        }
        if (wave == 0) DBG(0, rt() - tstart);
        asm volatile("s_waitcnt vmcnt(0)" : "+v"(fa[0][0]), "+v"(fa[0][1]), "+v"(fa[1][0]), "+v"(fa[1][1]), "+v"(fa[2][0]), "+v"(fa[2][1]), "+v"(fa[3][0]), "+v"(fa[3][1]),
                     "+v"(fa[4][0]), "+v"(fa[4][1]), "+v"(fa[5][0]), "+v"(fa[5][1]), "+v"(fa[6][0]), "+v"(fa[6][1]), "+v"(fa[7][0]), "+v"(fa[7][1]) :: "memory");
        if (__hip_atomic_load(tmo, __ATOMIC_RELAXED, __HIP_MEMORY_SCOPE_AGENT) == call || lds_ld(CTL + 3) != 0u) dead = 2;
        if (dead == 1 && lane == 0) __hip_atomic_store(tmo, call, __ATOMIC_RELAXED, __HIP_MEMORY_SCOPE_AGENT);
        const float poison = dead ? __builtin_nanf("") : 0.f;
#pragma unroll
        for (int rb = 0; rb < 2; ++rb)
#pragma unroll
            for (int cb = 0; cb < 4; ++cb)
#pragma unroll
                for (int r = 0; r < 16; ++r) {
                    const int row = mt * BM + wave * 64 + rb * 32 + (r & 3) + 8 * (r >> 2) + 4 * hi;
                    const int col = nt * BN + cb * 32 + l31;
                    out[(size_t)row * NN + col] = fmaxf(acc[rb][cb][r], 0.f) + poison;
                }
    } else if (wave < 6) {
        const int h = wave - 4, g = lane & 31, hh = lane >> 5;
        f4 r[PDEPTH][16];
        unsigned woff[2][4];
#pragma unroll
        for (int q = 0; q < 2; ++q)
#pragma unroll
            for (int i = 0; i < 4; ++i) { const int n = 4 * g + i, c = 4 * h + 2 * q + hh; woff[q][i] = n * 128 + ((c ^ fsw(n)) << 4); }
        const unsigned voff = (unsigned)((4 * g + (size_t)(32 * h + 8 * hh) * NN) * 4);
        const float* wtile0 = w + (size_t)nt * BN;
        auto issue = [&](int slot, int u) {
            const float* p = wtile0 + (size_t)ktile_of(u, rank) * BK * NN;
#pragma unroll
            for (int q = 0; q < 2; ++q)
#pragma unroll
                for (int j = 0; j < 8; ++j) {
                    const float* sp = p + (size_t)(16 * q + j) * NN;
                    asm volatile("global_load_dwordx4 %0, %1, %2" : "=v"(r[slot][q * 8 + j]) : "v"(voff), "s"(sp) : "memory");
                }
        };
#define PWAIT(slot) asm volatile("s_waitcnt vmcnt(%16)" : "+v"(r[slot][0]), "+v"(r[slot][1]), "+v"(r[slot][2]), "+v"(r[slot][3]), "+v"(r[slot][4]), "+v"(r[slot][5]), "+v"(r[slot][6]), "+v"(r[slot][7]), \
                                     "+v"(r[slot][8]), "+v"(r[slot][9]), "+v"(r[slot][10]), "+v"(r[slot][11]), "+v"(r[slot][12]), "+v"(r[slot][13]), "+v"(r[slot][14]), "+v"(r[slot][15]) : "n"(NW) : "memory")
        int sidx = 0;
        auto wr = [&](int slot, int u) {
            const int need = u - BSTAGES + 1;
            auto isfree = [&]() -> bool { return (int)lds_ld(CTL + 4) >= need && (int)lds_ld(CTL + 5) >= need && (int)lds_ld(CTL + 6) >= need && (int)lds_ld(CTL + 7) >= need; };
            if (!isfree()) {
                int sp = 0; const unsigned long long t0 = rt(); LGKM0();
                while (!isfree()) {
                    __builtin_amdgcn_s_sleep(2);
                    if ((++sp & 255) == 0) { if (lds_ld(CTL + 3) != 0u) break; if ((long long)(rt() - t0) > DEADLINE) { lds_st(CTL + 3, 1u); break; } }
                }
            }
            asm volatile("" ::: "memory");
            const unsigned soff = sidx * B_BYTES;
#pragma unroll
            for (int q = 0; q < 2; ++q)
#pragma unroll
                for (int i = 0; i < 4; ++i) {
                    h8 v;
#pragma unroll
                    for (int j = 0; j < 8; ++j) v[j] = (_Float16)r[slot][q * 8 + j][i];
                    *(LAS h8*)(L8 + soff + woff[q][i]) = v;
                }
            asm volatile("s_waitcnt lgkmcnt(0)" ::: "memory");
            lds_st(CTL + 1 + h, (unsigned)(u + 1));
            if (++sidx == BSTAGES) sidx = 0;
        };
        { int sp = 0; while (lds_ld(CTL + 9) == 0u && ++sp < 20000) __builtin_amdgcn_s_sleep(4); }
#pragma unroll
        for (int d = 0; d < PDEPTH; ++d) issue(d, d);
        constexpr int NFULL = (T_TILES / PDEPTH) * PDEPTH;
#pragma unroll 1
        for (int u0 = 0; u0 < NFULL; u0 += PDEPTH) {
#pragma unroll
            for (int d = 0; d < PDEPTH; ++d) {
                const int u = u0 + d;
                { constexpr int NW = 16 * (PDEPTH - 1); PWAIT(d); }
                wr(d, u);
                issue(d, min(u + PDEPTH, T_TILES - 1));
                SB();
            }
        }
#pragma unroll
        for (int d = 0; d < T_TILES - NFULL; ++d) { { constexpr int NW = 0; PWAIT(d); } wr(d, NFULL + d); }
        { constexpr int NW = 0; PWAIT(0); PWAIT(1); PWAIT(2); }
    } else if (wave == 6) {
        struct Piece { GAS unsigned char* dst; GAS unsigned* flag; const float* src; };
        auto piece = [&](int i) -> Piece {
            int dest, sb_, rg;
            if (i == 0) { dest = rank; sb_ = b >> 5; rg = b & 31; }
            else { const int id = b + 256 * (i - 1), grp = id >> 5; rg = id & 31; sb_ = grp / 7; dest = (rank + (grp - sb_ * 7) + 1) & 7; }
            GAS unsigned char* dws = (GAS unsigned char*)ws_all[dest];
            Piece pc;
            pc.dst = dws + WS_STAGE + (size_t)(call & 1) * SLOT_BYTES + ((size_t)(rg >> 3) * T_TILES + rank * 16 + 2 * sb_) * A_TILE_BYTES + (rg & 7) * 4096 + lane * 16;
            pc.flag = (GAS unsigned*)(dws + WS_FLAGS + (size_t)((rank * 8 + sb_) * 4 + (rg >> 3)) * 128) + (rg & 7);
            pc.src = x + (size_t)(dest * MLOC + rg * 32 + (lane & 31)) * KC + sb_ * 128 + (lane >> 5) * 8;
            return pc;
        };
        f4 cur[2][16];
        auto ldx = [&](int buf, const float* sp) {
#pragma unroll
            for (int j = 0; j < 8; ++j) { cur[buf][2 * j] = *(const f4*)(sp + j * 16); cur[buf][2 * j + 1] = *(const f4*)(sp + j * 16 + 4); }
        };
        auto cvst = [&](int buf, GAS unsigned char* dst) {
#pragma unroll
            for (int j = 0; j < 8; ++j) {
                h8 v;
#pragma unroll
                for (int e = 0; e < 4; ++e) { v[e] = (_Float16)cur[buf][2 * j][e]; v[4 + e] = (_Float16)cur[buf][2 * j + 1][e]; }
                asm volatile("global_store_dwordx4 %0, %1, off sc0 sc1\n\ts_nop 1" :: "v"(dst + (size_t)(j >> 2) * A_TILE_BYTES + (j & 3) * 1024), "v"(v) : "memory");
            }
        };
        Piece pcs[8];
#pragma unroll
        for (int i = 0; i < 8; ++i) pcs[i] = piece(i);
        DBG(8, rt() - tstart);
        ldx(0, pcs[0].src); ldx(1, pcs[1].src);
#pragma unroll
        for (int i = 0; i < 8; ++i) {
            cvst(i & 1, pcs[i].dst);
            if (i + 2 < 8) ldx(i & 1, pcs[i + 2].src);
            if (i == 0) {
                asm volatile("s_waitcnt vmcnt(16)" ::: "memory");
                if (lane == 0) __hip_atomic_store(pcs[0].flag, call, __ATOMIC_RELAXED, __HIP_MEMORY_SCOPE_SYSTEM);
                lds_st(CTL + 9, 1u);
                DBG(1, rt() - tstart);
            }
            if (i >= 2) {
                if (i + 2 < 8) asm volatile("s_waitcnt vmcnt(40)" ::: "memory"); else if (i == 6) asm volatile("s_waitcnt vmcnt(24)" ::: "memory"); else asm volatile("s_waitcnt vmcnt(8)" ::: "memory");
                if (lane == 0) __hip_atomic_store(pcs[i - 1].flag, call, __ATOMIC_RELAXED, __HIP_MEMORY_SCOPE_SYSTEM);
            }
        }
        asm volatile("s_waitcnt vmcnt(0)" ::: "memory");
        if (lane == 0) __hip_atomic_store(pcs[7].flag, call, __ATOMIC_RELAXED, __HIP_MEMORY_SCOPE_SYSTEM);
        DBG(2, rt() - tstart);
    } else {
        GAS unsigned* flags = (GAS unsigned*)(ws + WS_FLAGS);
        int un = 0;
        unsigned long long t0 = rt(); int sp = 0;
        while (un < NUNITS) {
            const int u = un + (lane >> 1);
            bool ok = true;
            if (u < NUNITS) {
                int p, s;
                if (u < 8) { p = rank; s = u; } else { const int v = u - 8; s = v / 7; p = (rank + (v - s * 7) + 1) & 7; }
                typedef unsigned u4v __attribute__((ext_vector_type(4)));
                u4v fv;
                asm volatile("global_load_dwordx4 %0, %1, off sc0 sc1\n\ts_waitcnt vmcnt(0)" : "=v"(fv) : "v"(flags + (size_t)((p * 8 + s) * 4 + mt) * 32 + (lane & 1) * 4) : "memory");
                ok = fv.x >= call && fv.y >= call && fv.z >= call && fv.w >= call;
            }
            const unsigned long long nr = __ballot(!ok);
            const int nready = nr ? (__builtin_ctzll(nr) >> 1) : 32;
            if (nready > 0) {
                __builtin_amdgcn_fence(__ATOMIC_ACQUIRE, "");
                asm volatile("s_waitcnt vmcnt(0)" ::: "memory");
                un = min(un + nready, NUNITS);
                lds_st(CTL + 0, (unsigned)(2 * un));
                t0 = rt(); sp = 0;
            } else {
                __builtin_amdgcn_s_sleep(2);
                if ((++sp & 63) == 0) {
                    if (lds_ld(CTL + 3) != 0u) break;
                    if (__hip_atomic_load(tmo, __ATOMIC_RELAXED, __HIP_MEMORY_SCOPE_AGENT) == call || (long long)(rt() - t0) > DEADLINE) {
                        __hip_atomic_store(tmo, call, __ATOMIC_RELAXED, __HIP_MEMORY_SCOPE_AGENT);
                        lds_st(CTL + 3, 1u);
                        break;
                    }
                }
            }
        }
    }
}

}

extern "C" void kernel_launch(int rank, int nranks, ncclComm_t comm, hipStream_t stream,
                              void* const* d_in, const int* in_sizes, int n_in,
                              void* d_out, int out_size,
                              const int* in_dtypes, int out_dtype,
                              void* const* d_out_all,
                              void* const* d_ws_all, size_t ws_bytes) {
    (void)comm; (void)in_sizes; (void)n_in; (void)out_size; (void)in_dtypes; (void)out_dtype; (void)d_out_all;
    if (nranks != NRANKS || (unsigned)rank >= (unsigned)NRANKS || ws_bytes < WS_NEED) {
        fprintf(stderr, "kernel_launch: unsupported configuration (nranks %d, rank %d, ws %zu)\n", nranks, rank, ws_bytes);
        return;
    }
    static bool attr_done[64] = {};
    int dev = 0;
    HIP_CHECK(hipGetDevice(&dev));
    if (!attr_done[dev & 63]) {
        HIP_CHECK(hipFuncSetAttribute((const void*)a2a_gemm_relu, hipFuncAttributeMaxDynamicSharedMemorySize, LDS_BYTES));
        attr_done[dev & 63] = true;
    }
    a2a_gemm_relu<<<NWG, 512, LDS_BYTES, stream>>>((const float*)d_in[0], (const float*)d_in[1], (float*)d_out,
                                                   (unsigned char* const*)d_ws_all, rank);
    HIP_CHECK(hipGetLastError());
}
```

```cpp
#include <hip/hip_runtime.h>
#include <rccl/rccl.h>
#include <cstdio>

#define HIP_CHECK(expr) do { hipError_t _e = (expr); if (_e != hipSuccess) { \
    fprintf(stderr, "HIP error %s at %s:%d\n", hipGetErrorString(_e), __FILE__, __LINE__); \
    return; } } while (0)

#define LAS __attribute__((address_space(3)))
#define GAS __attribute__((address_space(1)))
typedef _Float16 h8 __attribute__((ext_vector_type(8)));
typedef float f16v __attribute__((ext_vector_type(16)));
typedef float f4 __attribute__((ext_vector_type(4)));

namespace {

constexpr int NRANKS = 8;
constexpr int MLOC = 1024, NN = 8192, KK = 8192, KC = 1024;
constexpr int BM = 256, BN = 128, BK = 64;
constexpr int T_TILES = KK / BK;
constexpr int NUNITS = 64;
constexpr int BSTAGES = 4;
constexpr int A_TILE_BYTES = BM * BK * 2;
constexpr int B_BYTES = BN * BK * 2;
constexpr int CTL_OFF = BSTAGES * B_BYTES;
constexpr int LDS_BYTES = CTL_OFF + 64;
constexpr int PDEPTH = 3;
constexpr int NWG = 256;

constexpr size_t WS_TICKET = 0, WS_TMO = 128, WS_FLAGS = 4096;
constexpr size_t WS_STAGE = 1u << 20;
constexpr size_t SLOT_BYTES = (size_t)MLOC * KK * 2;
constexpr size_t WS_NEED = WS_STAGE + 2 * SLOT_BYTES;

constexpr long long DEADLINE = 40000000;

__device__ __forceinline__ unsigned long long rt() { return __builtin_amdgcn_s_memrealtime(); }
__device__ __forceinline__ unsigned lds_ld(LAS unsigned* p) { return __hip_atomic_load(p, __ATOMIC_RELAXED, __HIP_MEMORY_SCOPE_WORKGROUP); }
__device__ __forceinline__ void lds_st(LAS unsigned* p, unsigned v) { __hip_atomic_store(p, v, __ATOMIC_RELAXED, __HIP_MEMORY_SCOPE_WORKGROUP); }
__device__ __forceinline__ int fsw(int r) { return ((r >> 2) & 7) ^ ((r >> 1) & 1); }
__device__ __forceinline__ int ktile_of(int i, int rank) {
    if (i < 16) return rank * 16 + i;
    const int j = i - 16, pair = j >> 1, h = j & 1, s = pair / 7, q = pair - s * 7 + 1;
    return (((rank + q) & 7) << 4) + 2 * s + h;
}
#define SB() __builtin_amdgcn_sched_barrier(0)
#define LGKM0() __builtin_amdgcn_s_waitcnt(0xC07F)

__global__ __launch_bounds__(512, 2) void a2a_gemm_relu(const float* __restrict__ x, const float* __restrict__ w, float* __restrict__ out,
                                                        unsigned char* const* __restrict__ ws_all, int rank)
{
    extern __shared__ __attribute__((aligned(16))) unsigned char lds[];
    LAS unsigned char* L8 = (LAS unsigned char*)lds;
    LAS unsigned* CTL = (LAS unsigned*)(L8 + CTL_OFF);
    asm volatile("" :: "s"(x), "s"(w), "s"(out), "s"(ws_all), "s"(rank));
    const int tid = threadIdx.x, wave = __builtin_amdgcn_readfirstlane(tid >> 6), lane = tid & 63;
    const int b = blockIdx.x, xcd = b & 7, idx = b >> 3, mt = idx & 3, nt = xcd * 8 + (idx >> 2);
    GAS unsigned char* const ws = (GAS unsigned char*)ws_all[rank];
    if (tid < 16) CTL[tid] = 0u;
    __syncthreads();
    if (tid == 0) {
        const unsigned ticket = __hip_atomic_fetch_add((GAS unsigned*)(ws + WS_TICKET), 1u, __ATOMIC_RELAXED, __HIP_MEMORY_SCOPE_AGENT);
        CTL[8] = ticket / NWG + 1;
    }
    __syncthreads();
    const unsigned call = __builtin_amdgcn_readfirstlane(CTL[8]);
    GAS unsigned* const tmo = (GAS unsigned*)(ws + WS_TMO);

#define DBG(i, v)
    if (wave < 4) {
        const int l31 = lane & 31, hi = lane >> 5;
        const int f = fsw(l31);
        unsigned boff[4];
#pragma unroll
        for (int kk = 0; kk < 4; ++kk) boff[kk] = l31 * 128 + (((2 * kk + hi) ^ f) << 4);
        f16v acc[2][4];
#pragma unroll
        for (int i = 0; i < 2; ++i)
#pragma unroll
            for (int j = 0; j < 4; ++j)
#pragma unroll
                for (int r = 0; r < 16; ++r) acc[i][j][r] = 0.f;
        h8 fa[8][2];
        h8 fb[2][4];
        unsigned soff = 0;
        int dead = 0;
        const unsigned voff = lane * 16;
        const GAS unsigned char* abase = ws + WS_STAGE + (size_t)(call & 1) * SLOT_BYTES + (size_t)mt * T_TILES * A_TILE_BYTES + wave * 8192;
#define ALOAD(slot, ptile, kk) do { \
            asm volatile("global_load_dwordx4 %0, %1, %2 offset:%3" : "=v"(fa[slot][0]) : "v"(voff), "s"(ptile), "n"((kk) * 1024) : "memory"); \
            asm volatile("global_load_dwordx4 %0, %1, %2 offset:%3" : "=v"(fa[slot][1]) : "v"(voff), "s"((ptile) + 4096), "n"((kk) * 1024) : "memory"); } while (0)
#define AWAIT(slot, n) asm volatile("s_waitcnt vmcnt(%2)" : "+v"(fa[slot][0]), "+v"(fa[slot][1]) : "n"(n) : "memory")
#define MM(slot, set, rb, cb) acc[rb][cb] = __builtin_amdgcn_mfma_f32_32x32x16_f16(fa[slot][rb], fb[set][cb], acc[rb][cb], 0, 0, 0)
#define BRD(set, so, kk, cb) fb[set][cb] = *(const LAS h8*)(L8 + (so) + boff[kk] + (cb) * 4096)
        auto slow_wait = [&](int word0, int nwords, unsigned need) {
            const unsigned long long t0 = rt(); LGKM0(); int sp = 0;
            for (;;) {
                bool ok = true;
                for (int i = 0; i < nwords; ++i) ok = ok && (__builtin_amdgcn_readfirstlane(lds_ld(CTL + word0 + i)) >= need);
                if (ok) break;
                if ((++sp & 255) == 0) {
                    if (lds_ld(CTL + 3) != 0u) { dead = 2; break; }
                    if ((long long)(rt() - t0) > DEADLINE) { dead = 1; lds_st(CTL + 3, 1u); break; }
                }
            }
            asm volatile("" ::: "memory");
        };
        auto atile = [&](int t) -> const GAS unsigned char* { return abase + (size_t)ktile_of(min(t, T_TILES - 1), rank) * A_TILE_BYTES; };
        if (__builtin_amdgcn_readfirstlane(lds_ld(CTL + 0)) < 2u) slow_wait(0, 1, 2u);
        if (wave == 0) DBG(7, rt() - tstart);
        { const GAS unsigned char* p0 = atile(0); const GAS unsigned char* p1 = atile(1);
          ALOAD(0, p0, 0); ALOAD(1, p0, 1); ALOAD(2, p0, 2); ALOAD(3, p0, 3); ALOAD(4, p1, 0); ALOAD(5, p1, 1); ALOAD(6, p1, 2); }
        if (__builtin_amdgcn_readfirstlane(lds_ld(CTL + 1)) < 1u || __builtin_amdgcn_readfirstlane(lds_ld(CTL + 2)) < 1u) slow_wait(1, 2, 1u);
#pragma unroll
        for (int cb = 0; cb < 4; ++cb) BRD(0, soff, 0, cb);
        unsigned rdy = lds_ld(CTL + 0);
        SB();
#define STEPI(slot, set, so_n, kk_n, aslot, aptr, akk) do { \
            MM(slot, set, 0, 0); MM(slot, set, 1, 0); BRD(1 - (set), so_n, kk_n, 0); SB(); \
            MM(slot, set, 0, 1); BRD(1 - (set), so_n, kk_n, 1); MM(slot, set, 1, 1); BRD(1 - (set), so_n, kk_n, 2); SB(); \
            MM(slot, set, 0, 2); BRD(1 - (set), so_n, kk_n, 3); MM(slot, set, 1, 2); SB(); \
            ALOAD(aslot, aptr, akk); SB(); \
            MM(slot, set, 0, 3); MM(slot, set, 1, 3); SB(); } while (0)
#define TILE(PAR, t) do { \
            const GAS unsigned char* pn = atile((t) + 1); const GAS unsigned char* pnn = atile((t) + 2); \
            { const unsigned needA = (unsigned)min((t) + 3, T_TILES); if (__builtin_amdgcn_readfirstlane(rdy) < needA) slow_wait(0, 1, needA); } \
            AWAIT((PAR) * 4 + 0, 12); STEPI((PAR) * 4 + 0, 0, soff, 1, (1 - (PAR)) * 4 + 3, pn, 3); \
            AWAIT((PAR) * 4 + 1, 12); STEPI((PAR) * 4 + 1, 1, soff, 2, (PAR) * 4 + 0, pnn, 0); \
            AWAIT((PAR) * 4 + 2, 12); STEPI((PAR) * 4 + 2, 0, soff, 3, (PAR) * 4 + 1, pnn, 1); \
            const unsigned pb0 = lds_ld(CTL + 1), pb1 = lds_ld(CTL + 2); rdy = lds_ld(CTL + 0); \
            AWAIT((PAR) * 4 + 3, 12); \
            asm volatile("" ::: "memory"); LGKM0(); \
            lds_st(CTL + 4 + wave, (unsigned)((t) + 1)); \
            unsigned nso = soff + B_BYTES; if (nso == BSTAGES * B_BYTES) nso = 0; \
            if ((t) + 1 < T_TILES && (__builtin_amdgcn_readfirstlane(pb0) < (unsigned)((t) + 2) || __builtin_amdgcn_readfirstlane(pb1) < (unsigned)((t) + 2))) slow_wait(1, 2, (unsigned)((t) + 2)); \
            asm volatile("" ::: "memory"); \
            STEPI((PAR) * 4 + 3, 1, nso, 0, (PAR) * 4 + 2, pnn, 2); \
            soff = nso; } while (0)
#pragma unroll 1
        for (int t = 0; t < T_TILES; t += 2) {
            TILE(0, t);
            TILE(1, t + 1);
            if (dead) break;
        }
        if (wave == 0) DBG(0, rt() - tstart);
        asm volatile("s_waitcnt vmcnt(0)" : "+v"(fa[0][0]), "+v"(fa[0][1]), "+v"(fa[1][0]), "+v"(fa[1][1]), "+v"(fa[2][0]), "+v"(fa[2][1]), "+v"(fa[3][0]), "+v"(fa[3][1]),
                     "+v"(fa[4][0]), "+v"(fa[4][1]), "+v"(fa[5][0]), "+v"(fa[5][1]), "+v"(fa[6][0]), "+v"(fa[6][1]), "+v"(fa[7][0]), "+v"(fa[7][1]) :: "memory");
        if (__hip_atomic_load(tmo, __ATOMIC_RELAXED, __HIP_MEMORY_SCOPE_AGENT) == call || lds_ld(CTL + 3) != 0u) dead = 2;
        if (dead == 1 && lane == 0) __hip_atomic_store(tmo, call, __ATOMIC_RELAXED, __HIP_MEMORY_SCOPE_AGENT);
        const float poison = dead ? __builtin_nanf("") : 0.f;
#pragma unroll
        for (int rb = 0; rb < 2; ++rb)
#pragma unroll
            for (int cb = 0; cb < 4; ++cb)
#pragma unroll
                for (int r = 0; r < 16; ++r) {
                    const int row = mt * BM + wave * 64 + rb * 32 + (r & 3) + 8 * (r >> 2) + 4 * hi;
                    const int col = nt * BN + cb * 32 + l31;
                    out[(size_t)row * NN + col] = fmaxf(acc[rb][cb][r], 0.f) + poison;
                }
    } else if (wave < 6) {
        const int h = wave - 4, g = lane & 31, hh = lane >> 5;
        f4 r[PDEPTH][16];
        unsigned woff[2][4];
#pragma unroll
        for (int q = 0; q < 2; ++q)
#pragma unroll
            for (int i = 0; i < 4; ++i) { const int n = 4 * g + i, c = 4 * h + 2 * q + hh; woff[q][i] = n * 128 + ((c ^ fsw(n)) << 4); }
        const unsigned voff = (unsigned)((4 * g + (size_t)(32 * h + 8 * hh) * NN) * 4);
        const float* wtile0 = w + (size_t)nt * BN;
        auto issue = [&](int slot, int u) {
            const float* p = wtile0 + (size_t)ktile_of(u, rank) * BK * NN;
#pragma unroll
            for (int q = 0; q < 2; ++q)
#pragma unroll
                for (int j = 0; j < 8; ++j) {
                    const float* sp = p + (size_t)(16 * q + j) * NN;
                    asm volatile("global_load_dwordx4 %0, %1, %2" : "=v"(r[slot][q * 8 + j]) : "v"(voff), "s"(sp) : "memory");
                }
        };
#define PWAIT(slot) asm volatile("s_waitcnt vmcnt(%16)" : "+v"(r[slot][0]), "+v"(r[slot][1]), "+v"(r[slot][2]), "+v"(r[slot][3]), "+v"(r[slot][4]), "+v"(r[slot][5]), "+v"(r[slot][6]), "+v"(r[slot][7]), \
                                     "+v"(r[slot][8]), "+v"(r[slot][9]), "+v"(r[slot][10]), "+v"(r[slot][11]), "+v"(r[slot][12]), "+v"(r[slot][13]), "+v"(r[slot][14]), "+v"(r[slot][15]) : "n"(NW) : "memory")
        int sidx = 0;
        auto wr = [&](int slot, int u) {
            const int need = u - BSTAGES + 1;
            auto isfree = [&]() -> bool { return (int)lds_ld(CTL + 4) >= need && (int)lds_ld(CTL + 5) >= need && (int)lds_ld(CTL + 6) >= need && (int)lds_ld(CTL + 7) >= need; };
            if (!isfree()) {
                int sp = 0; const unsigned long long t0 = rt(); LGKM0();
                while (!isfree()) {
                    __builtin_amdgcn_s_sleep(2);
                    if ((++sp & 255) == 0) { if (lds_ld(CTL + 3) != 0u) break; if ((long long)(rt() - t0) > DEADLINE) { lds_st(CTL + 3, 1u); break; } }
                }
            }
            asm volatile("" ::: "memory");
            const unsigned soff = sidx * B_BYTES;
#pragma unroll
            for (int q = 0; q < 2; ++q)
#pragma unroll
                for (int i = 0; i < 4; ++i) {
                    h8 v;
#pragma unroll
                    for (int j = 0; j < 8; ++j) v[j] = (_Float16)r[slot][q * 8 + j][i];
                    *(LAS h8*)(L8 + soff + woff[q][i]) = v;
                }
            asm volatile("s_waitcnt lgkmcnt(0)" ::: "memory");
            lds_st(CTL + 1 + h, (unsigned)(u + 1));
            if (++sidx == BSTAGES) sidx = 0;
        };
        { int sp = 0; while (lds_ld(CTL + 9) == 0u && ++sp < 20000) __builtin_amdgcn_s_sleep(4); }
#pragma unroll
        for (int d = 0; d < PDEPTH; ++d) issue(d, d);
        constexpr int NFULL = (T_TILES / PDEPTH) * PDEPTH;
#pragma unroll 1
        for (int u0 = 0; u0 < NFULL; u0 += PDEPTH) {
#pragma unroll
            for (int d = 0; d < PDEPTH; ++d) {
                const int u = u0 + d;
                { constexpr int NW = 16 * (PDEPTH - 1); PWAIT(d); }
                wr(d, u);
                issue(d, min(u + PDEPTH, T_TILES - 1));
                SB();
            }
        }
#pragma unroll
        for (int d = 0; d < T_TILES - NFULL; ++d) { { constexpr int NW = 0; PWAIT(d); } wr(d, NFULL + d); }
        { constexpr int NW = 0; PWAIT(0); PWAIT(1); PWAIT(2); }
    } else if (wave == 6) {
        struct Piece { GAS unsigned char* dst; GAS unsigned* flag; const float* src; };
        auto piece = [&](int i) -> Piece {
            int dest, sb_, rg;
            if (i == 0) { dest = rank; sb_ = b >> 5; rg = b & 31; }
            else { const int id = b + 256 * (i - 1), grp = id >> 5; rg = id & 31; sb_ = grp / 7; dest = (rank + (grp - sb_ * 7) + 1) & 7; }
            GAS unsigned char* dws = (GAS unsigned char*)ws_all[dest];
            Piece pc;
            pc.dst = dws + WS_STAGE + (size_t)(call & 1) * SLOT_BYTES + ((size_t)(rg >> 3) * T_TILES + rank * 16 + 2 * sb_) * A_TILE_BYTES + (rg & 7) * 4096 + lane * 16;
            pc.flag = (GAS unsigned*)(dws + WS_FLAGS + (size_t)((rank * 8 + sb_) * 4 + (rg >> 3)) * 128) + (rg & 7);
            pc.src = x + (size_t)(dest * MLOC + rg * 32 + (lane & 31)) * KC + sb_ * 128 + (lane >> 5) * 8;
            return pc;
        };
        f4 cur[2][16];
        auto ldx = [&](int buf, const float* sp) {
#pragma unroll
            for (int j = 0; j < 8; ++j) { cur[buf][2 * j] = *(const f4*)(sp + j * 16); cur[buf][2 * j + 1] = *(const f4*)(sp + j * 16 + 4); }
        };
#define SWIN 1
        auto cvst = [&](int buf, GAS unsigned char* dst, bool remote) {
#pragma unroll
            for (int j = 0; j < 8; ++j) {
                h8 v;
#pragma unroll
                for (int e = 0; e < 4; ++e) { v[e] = (_Float16)cur[buf][2 * j][e]; v[4 + e] = (_Float16)cur[buf][2 * j + 1][e]; }
                asm volatile("global_store_dwordx4 %0, %1, off sc0 sc1\n\ts_nop 1" :: "v"(dst + (size_t)(j >> 2) * A_TILE_BYTES + (j & 3) * 1024), "v"(v) : "memory");
                if (SWIN < 8 && remote && (j + 1) % SWIN == 0 && j != 7) asm volatile("s_waitcnt vmcnt(0)" ::: "memory");
            }
        };
        Piece pcs[8];
#pragma unroll
        for (int i = 0; i < 8; ++i) pcs[i] = piece(i);
        DBG(8, rt() - tstart);
        ldx(0, pcs[0].src); ldx(1, pcs[1].src);
#pragma unroll
        for (int i = 0; i < 8; ++i) {
            cvst(i & 1, pcs[i].dst, i > 0);
            if (i + 2 < 8) { ldx(i & 1, pcs[i + 2].src); asm volatile("s_waitcnt vmcnt(16)" ::: "memory"); }
            else asm volatile("s_waitcnt vmcnt(0)" ::: "memory");
            if (lane == 0) __hip_atomic_store(pcs[i].flag, call, __ATOMIC_RELAXED, __HIP_MEMORY_SCOPE_SYSTEM);
            if (i == 0) { lds_st(CTL + 9, 1u); DBG(1, rt() - tstart); }
        }
        DBG(2, rt() - tstart);
    } else {
        GAS unsigned* flags = (GAS unsigned*)(ws + WS_FLAGS);
        int un = 0;
        unsigned long long t0 = rt(); int sp = 0;
        while (un < NUNITS) {
            const int u = un + (lane >> 1);
            bool ok = true;
            if (u < NUNITS) {
                int p, s;
                if (u < 8) { p = rank; s = u; } else { const int v = u - 8; s = v / 7; p = (rank + (v - s * 7) + 1) & 7; }
                typedef unsigned u4v __attribute__((ext_vector_type(4)));
                u4v fv;
                asm volatile("global_load_dwordx4 %0, %1, off sc0 sc1\n\ts_waitcnt vmcnt(0)" : "=v"(fv) : "v"(flags + (size_t)((p * 8 + s) * 4 + mt) * 32 + (lane & 1) * 4) : "memory");
                ok = fv.x >= call && fv.y >= call && fv.z >= call && fv.w >= call;
            }
            const unsigned long long nr = __ballot(!ok);
            const int nready = nr ? (__builtin_ctzll(nr) >> 1) : 32;
            if (nready > 0) {
                un = min(un + nready, NUNITS);
                lds_st(CTL + 0, (unsigned)(2 * un));
                t0 = rt(); sp = 0;
            } else {
                __builtin_amdgcn_s_sleep(2);
                if ((++sp & 63) == 0) {
                    if (lds_ld(CTL + 3) != 0u) break;
                    if (__hip_atomic_load(tmo, __ATOMIC_RELAXED, __HIP_MEMORY_SCOPE_AGENT) == call || (long long)(rt() - t0) > DEADLINE) {
                        __hip_atomic_store(tmo, call, __ATOMIC_RELAXED, __HIP_MEMORY_SCOPE_AGENT);
                        lds_st(CTL + 3, 1u);
                        break;
                    }
                }
            }
        }
    }
}

}

extern "C" void kernel_launch(int rank, int nranks, ncclComm_t comm, hipStream_t stream,
                              void* const* d_in, const int* in_sizes, int n_in,
                              void* d_out, int out_size,
                              const int* in_dtypes, int out_dtype,
                              void* const* d_out_all,
                              void* const* d_ws_all, size_t ws_bytes) {
    (void)comm; (void)in_sizes; (void)n_in; (void)out_size; (void)in_dtypes; (void)out_dtype; (void)d_out_all;
    if (nranks != NRANKS || (unsigned)rank >= (unsigned)NRANKS || ws_bytes < WS_NEED) {
        fprintf(stderr, "kernel_launch: unsupported configuration (nranks %d, rank %d, ws %zu)\n", nranks, rank, ws_bytes);
        return;
    }
    static bool attr_done[64] = {};
    int dev = 0;
    HIP_CHECK(hipGetDevice(&dev));
    if (!attr_done[dev & 63]) {
        HIP_CHECK(hipFuncSetAttribute((const void*)a2a_gemm_relu, hipFuncAttributeMaxDynamicSharedMemorySize, LDS_BYTES));
        attr_done[dev & 63] = true;
    }
    a2a_gemm_relu<<<NWG, 512, LDS_BYTES, stream>>>((const float*)d_in[0], (const float*)d_in[1], (float*)d_out,
                                                   (unsigned char* const*)d_ws_all, rank);
    HIP_CHECK(hipGetLastError());
}
```

```cpp
#include <hip/hip_runtime.h>
#include <rccl/rccl.h>
#include <cstdio>

#define HIP_CHECK(expr) do { hipError_t _e = (expr); if (_e != hipSuccess) { \
    fprintf(stderr, "HIP error %s at %s:%d\n", hipGetErrorString(_e), __FILE__, __LINE__); \
    return; } } while (0)

#define LAS __attribute__((address_space(3)))
#define GAS __attribute__((address_space(1)))
typedef _Float16 h8 __attribute__((ext_vector_type(8)));
typedef float f16v __attribute__((ext_vector_type(16)));
typedef float f4 __attribute__((ext_vector_type(4)));

namespace {

constexpr int NRANKS = 8;
constexpr int MLOC = 1024, NN = 8192, KK = 8192, KC = 1024;
constexpr int BM = 256, BN = 128, BK = 64;
constexpr int T_TILES = KK / BK;
constexpr int NUNITS = 64;
constexpr int BSTAGES = 4;
constexpr int A_TILE_BYTES = BM * BK * 2;
constexpr int B_BYTES = BN * BK * 2;
constexpr int CTL_OFF = BSTAGES * B_BYTES;
constexpr int LDS_BYTES = CTL_OFF + 64;
constexpr int PDEPTH = 3;
constexpr int NWG = 256;

constexpr size_t WS_TICKET = 0, WS_TMO = 128, WS_FLAGS = 4096;
constexpr size_t WS_STAGE = 1u << 20;
constexpr size_t SLOT_BYTES = (size_t)MLOC * KK * 2;
constexpr size_t WS_NEED = WS_STAGE + 2 * SLOT_BYTES;

constexpr long long DEADLINE = 40000000;

__device__ __forceinline__ unsigned long long rt() { return __builtin_amdgcn_s_memrealtime(); }
__device__ __forceinline__ unsigned lds_ld(LAS unsigned* p) { return __hip_atomic_load(p, __ATOMIC_RELAXED, __HIP_MEMORY_SCOPE_WORKGROUP); }
__device__ __forceinline__ void lds_st(LAS unsigned* p, unsigned v) { __hip_atomic_store(p, v, __ATOMIC_RELAXED, __HIP_MEMORY_SCOPE_WORKGROUP); }
__device__ __forceinline__ int fsw(int r) { return ((r >> 2) & 7) ^ ((r >> 1) & 1); }
__device__ __forceinline__ int ktile_of(int i, int rank) {
    if (i < 16) return rank * 16 + i;
    const int j = i - 16, pair = j >> 1, h = j & 1, s = pair / 7, q = pair - s * 7 + 1;
    return (((rank + q) & 7) << 4) + 2 * s + h;
}
#define SB() __builtin_amdgcn_sched_barrier(0)
#define LGKM0() __builtin_amdgcn_s_waitcnt(0xC07F)

__global__ __launch_bounds__(512, 2) void a2a_gemm_relu(const float* __restrict__ x, const float* __restrict__ w, float* __restrict__ out,
                                                        unsigned char* const* __restrict__ ws_all, int rank)
{
    extern __shared__ __attribute__((aligned(16))) unsigned char lds[];
    LAS unsigned char* L8 = (LAS unsigned char*)lds;
    LAS unsigned* CTL = (LAS unsigned*)(L8 + CTL_OFF);
    asm volatile("" :: "s"(x), "s"(w), "s"(out), "s"(ws_all), "s"(rank));
    const int tid = threadIdx.x, wave = __builtin_amdgcn_readfirstlane(tid >> 6), lane = tid & 63;
    const int b = blockIdx.x, xcd = b & 7, idx = b >> 3, mt = idx & 3, nt = xcd * 8 + (idx >> 2);
    GAS unsigned char* const ws = (GAS unsigned char*)ws_all[rank];
    if (tid < 16) CTL[tid] = 0u;
    __syncthreads();
    if (tid == 0) {
        const unsigned ticket = __hip_atomic_fetch_add((GAS unsigned*)(ws + WS_TICKET), 1u, __ATOMIC_RELAXED, __HIP_MEMORY_SCOPE_AGENT);
        CTL[8] = ticket / NWG + 1;
    }
    __syncthreads();
    const unsigned call = __builtin_amdgcn_readfirstlane(CTL[8]);
    GAS unsigned* const tmo = (GAS unsigned*)(ws + WS_TMO);

    if (wave < 4) {
        const int l31 = lane & 31, hi = lane >> 5;
        const int f = fsw(l31);
        unsigned boff[4];
#pragma unroll
        for (int kk = 0; kk < 4; ++kk) boff[kk] = l31 * 128 + (((2 * kk + hi) ^ f) << 4);
        f16v acc[2][4];
#pragma unroll
        for (int i = 0; i < 2; ++i)
#pragma unroll
            for (int j = 0; j < 4; ++j)
#pragma unroll
                for (int r = 0; r < 16; ++r) acc[i][j][r] = 0.f;
        h8 fa[8][2];
        h8 fb[2][4];
        unsigned soff = 0;
        int dead = 0;
        const unsigned voff = lane * 16;
        const GAS unsigned char* abase = ws + WS_STAGE + (size_t)(call & 1) * SLOT_BYTES + (size_t)mt * T_TILES * A_TILE_BYTES + wave * 8192;
#define ALOAD(slot, ptile, kk) do { \
            asm volatile("global_load_dwordx4 %0, %1, %2 offset:%3" : "=v"(fa[slot][0]) : "v"(voff), "s"(ptile), "n"((kk) * 1024) : "memory"); \
            asm volatile("global_load_dwordx4 %0, %1, %2 offset:%3" : "=v"(fa[slot][1]) : "v"(voff), "s"((ptile) + 4096), "n"((kk) * 1024) : "memory"); } while (0)
#define AWAIT(slot, n) asm volatile("s_waitcnt vmcnt(%2)" : "+v"(fa[slot][0]), "+v"(fa[slot][1]) : "n"(n) : "memory")
#define MM(slot, set, rb, cb) acc[rb][cb] = __builtin_amdgcn_mfma_f32_32x32x16_f16(fa[slot][rb], fb[set][cb], acc[rb][cb], 0, 0, 0)
#define BRD(set, so, kk, cb) fb[set][cb] = *(const LAS h8*)(L8 + (so) + boff[kk] + (cb) * 4096)
        auto slow_wait = [&](int word0, int nwords, unsigned need) {
            const unsigned long long t0 = rt(); LGKM0(); int sp = 0;
            for (;;) {
                bool ok = true;
                for (int i = 0; i < nwords; ++i) ok = ok && (__builtin_amdgcn_readfirstlane(lds_ld(CTL + word0 + i)) >= need);
                if (ok) break;
                if ((++sp & 255) == 0) {
                    if (lds_ld(CTL + 3) != 0u) { dead = 2; break; }
                    if ((long long)(rt() - t0) > DEADLINE) { dead = 1; lds_st(CTL + 3, 1u); break; }
                }
            }
            asm volatile("" ::: "memory");
        };
        auto atile = [&](int t) -> const GAS unsigned char* { return abase + (size_t)ktile_of(min(t, T_TILES - 1), rank) * A_TILE_BYTES; };
        if (__builtin_amdgcn_readfirstlane(lds_ld(CTL + 0)) < 2u) slow_wait(0, 1, 2u);
        { const GAS unsigned char* p0 = atile(0); const GAS unsigned char* p1 = atile(1);
          ALOAD(0, p0, 0); ALOAD(1, p0, 1); ALOAD(2, p0, 2); ALOAD(3, p0, 3); ALOAD(4, p1, 0); ALOAD(5, p1, 1); ALOAD(6, p1, 2); }
        if (__builtin_amdgcn_readfirstlane(lds_ld(CTL + 1)) < 1u || __builtin_amdgcn_readfirstlane(lds_ld(CTL + 2)) < 1u) slow_wait(1, 2, 1u);
#pragma unroll
        for (int cb = 0; cb < 4; ++cb) BRD(0, soff, 0, cb);
        unsigned rdy = lds_ld(CTL + 0);
        SB();
#define STEPI(slot, set, so_n, kk_n, aslot, aptr, akk) do { \
            MM(slot, set, 0, 0); MM(slot, set, 1, 0); BRD(1 - (set), so_n, kk_n, 0); SB(); \
            MM(slot, set, 0, 1); BRD(1 - (set), so_n, kk_n, 1); MM(slot, set, 1, 1); BRD(1 - (set), so_n, kk_n, 2); SB(); \
            MM(slot, set, 0, 2); BRD(1 - (set), so_n, kk_n, 3); MM(slot, set, 1, 2); SB(); \
            ALOAD(aslot, aptr, akk); SB(); \
            MM(slot, set, 0, 3); MM(slot, set, 1, 3); SB(); } while (0)
#define TILE(PAR, t) do { \
            const GAS unsigned char* pn = atile((t) + 1); const GAS unsigned char* pnn = atile((t) + 2); \
            { const unsigned needA = (unsigned)min((t) + 3, T_TILES); if (__builtin_amdgcn_readfirstlane(rdy) < needA) slow_wait(0, 1, needA); } \
            AWAIT((PAR) * 4 + 0, 12); STEPI((PAR) * 4 + 0, 0, soff, 1, (1 - (PAR)) * 4 + 3, pn, 3); \
            AWAIT((PAR) * 4 + 1, 12); STEPI((PAR) * 4 + 1, 1, soff, 2, (PAR) * 4 + 0, pnn, 0); \
            AWAIT((PAR) * 4 + 2, 12); STEPI((PAR) * 4 + 2, 0, soff, 3, (PAR) * 4 + 1, pnn, 1); \
            const unsigned pb0 = lds_ld(CTL + 1), pb1 = lds_ld(CTL + 2); rdy = lds_ld(CTL + 0); \
            AWAIT((PAR) * 4 + 3, 12); \
            asm volatile("" ::: "memory"); LGKM0(); \
            lds_st(CTL + 4 + wave, (unsigned)((t) + 1)); \
            unsigned nso = soff + B_BYTES; if (nso == BSTAGES * B_BYTES) nso = 0; \
            if ((t) + 1 < T_TILES && (__builtin_amdgcn_readfirstlane(pb0) < (unsigned)((t) + 2) || __builtin_amdgcn_readfirstlane(pb1) < (unsigned)((t) + 2))) slow_wait(1, 2, (unsigned)((t) + 2)); \
            asm volatile("" ::: "memory"); \
            STEPI((PAR) * 4 + 3, 1, nso, 0, (PAR) * 4 + 2, pnn, 2); \
            soff = nso; } while (0)
#pragma unroll 1
        for (int t = 0; t < T_TILES; t += 2) {
            TILE(0, t);
            TILE(1, t + 1);
            if (dead) break;
        }
        asm volatile("s_waitcnt vmcnt(0)" : "+v"(fa[0][0]), "+v"(fa[0][1]), "+v"(fa[1][0]), "+v"(fa[1][1]), "+v"(fa[2][0]), "+v"(fa[2][1]), "+v"(fa[3][0]), "+v"(fa[3][1]),
                     "+v"(fa[4][0]), "+v"(fa[4][1]), "+v"(fa[5][0]), "+v"(fa[5][1]), "+v"(fa[6][0]), "+v"(fa[6][1]), "+v"(fa[7][0]), "+v"(fa[7][1]) :: "memory");
        if (__hip_atomic_load(tmo, __ATOMIC_RELAXED, __HIP_MEMORY_SCOPE_AGENT) == call || lds_ld(CTL + 3) != 0u) dead = 2;
        if (dead == 1 && lane == 0) __hip_atomic_store(tmo, call, __ATOMIC_RELAXED, __HIP_MEMORY_SCOPE_AGENT);
        const float poison = dead ? __builtin_nanf("") : 0.f;
#pragma unroll
        for (int rb = 0; rb < 2; ++rb)
#pragma unroll
            for (int cb = 0; cb < 4; ++cb)
#pragma unroll
                for (int r = 0; r < 16; ++r) {
                    const int row = mt * BM + wave * 64 + rb * 32 + (r & 3) + 8 * (r >> 2) + 4 * hi;
                    const int col = nt * BN + cb * 32 + l31;
                    out[(size_t)row * NN + col] = fmaxf(acc[rb][cb][r], 0.f) + poison;
                }
    } else if (wave < 6) {
        const int h = wave - 4, g = lane & 31, hh = lane >> 5;
        f4 r[PDEPTH][16];
        unsigned woff[2][4];
#pragma unroll
        for (int q = 0; q < 2; ++q)
#pragma unroll
            for (int i = 0; i < 4; ++i) { const int n = 4 * g + i, c = 4 * h + 2 * q + hh; woff[q][i] = n * 128 + ((c ^ fsw(n)) << 4); }
        const unsigned voff = (unsigned)((4 * g + (size_t)(32 * h + 8 * hh) * NN) * 4);
        const float* wtile0 = w + (size_t)nt * BN;
        auto issue = [&](int slot, int u) {
            const float* p = wtile0 + (size_t)ktile_of(u, rank) * BK * NN;
#pragma unroll
            for (int q = 0; q < 2; ++q)
#pragma unroll
                for (int j = 0; j < 8; ++j) {
                    const float* sp = p + (size_t)(16 * q + j) * NN;
                    asm volatile("global_load_dwordx4 %0, %1, %2" : "=v"(r[slot][q * 8 + j]) : "v"(voff), "s"(sp) : "memory");
                }
        };
#define PWAIT(slot) asm volatile("s_waitcnt vmcnt(%16)" : "+v"(r[slot][0]), "+v"(r[slot][1]), "+v"(r[slot][2]), "+v"(r[slot][3]), "+v"(r[slot][4]), "+v"(r[slot][5]), "+v"(r[slot][6]), "+v"(r[slot][7]), \
                                     "+v"(r[slot][8]), "+v"(r[slot][9]), "+v"(r[slot][10]), "+v"(r[slot][11]), "+v"(r[slot][12]), "+v"(r[slot][13]), "+v"(r[slot][14]), "+v"(r[slot][15]) : "n"(NW) : "memory")
        int sidx = 0;
        auto wr = [&](int slot, int u) {
            const int need = u - BSTAGES + 1;
            auto isfree = [&]() -> bool { const int f0 = lds_ld(CTL + 4), f1 = lds_ld(CTL + 5), f2 = lds_ld(CTL + 6), f3 = lds_ld(CTL + 7); return min(min(f0, f1), min(f2, f3)) >= need; };
            if (!isfree()) {
                int sp = 0; const unsigned long long t0 = rt(); LGKM0();
                while (!isfree()) {
                    __builtin_amdgcn_s_sleep(2);
                    if ((++sp & 255) == 0) { if (lds_ld(CTL + 3) != 0u) break; if ((long long)(rt() - t0) > DEADLINE) { lds_st(CTL + 3, 1u); break; } }
                }
            }
            asm volatile("" ::: "memory");
            const unsigned soff = sidx * B_BYTES;
#pragma unroll
            for (int q = 0; q < 2; ++q)
#pragma unroll
                for (int i = 0; i < 4; ++i) {
                    h8 v;
#pragma unroll
                    for (int j = 0; j < 8; ++j) v[j] = (_Float16)r[slot][q * 8 + j][i];
                    *(LAS h8*)(L8 + soff + woff[q][i]) = v;
                }
            asm volatile("" ::: "memory");
            lds_st(CTL + 1 + h, (unsigned)(u + 1));
            if (++sidx == BSTAGES) sidx = 0;
        };
        { int sp = 0; while (lds_ld(CTL + 9) == 0u && ++sp < 20000) __builtin_amdgcn_s_sleep(4); }
#pragma unroll
        for (int d = 0; d < PDEPTH; ++d) issue(d, d);
        constexpr int NFULL = (T_TILES / PDEPTH) * PDEPTH;
#pragma unroll 1
        for (int u0 = 0; u0 < NFULL; u0 += PDEPTH) {
#pragma unroll
            for (int d = 0; d < PDEPTH; ++d) {
                const int u = u0 + d;
                { constexpr int NW = 16 * (PDEPTH - 1); PWAIT(d); }
                wr(d, u);
                issue(d, min(u + PDEPTH, T_TILES - 1));
                SB();
            }
        }
#pragma unroll
        for (int d = 0; d < T_TILES - NFULL; ++d) { { constexpr int NW = 0; PWAIT(d); } wr(d, NFULL + d); }
        { constexpr int NW = 0; PWAIT(0); PWAIT(1); PWAIT(2); }
    } else if (wave == 6) {
        struct Piece { GAS unsigned char* dst; GAS unsigned* flag; const float* src; };
        auto piece = [&](int i) -> Piece {
            int dest, sb_, rg;
            if (i == 0) { dest = rank; sb_ = b >> 5; rg = b & 31; }
            else { const int id = b + 256 * (i - 1), grp = id >> 5; rg = id & 31; sb_ = grp / 7; dest = (rank + (grp - sb_ * 7) + 1) & 7; }
            GAS unsigned char* dws = (GAS unsigned char*)ws_all[dest];
            Piece pc;
            pc.dst = dws + WS_STAGE + (size_t)(call & 1) * SLOT_BYTES + ((size_t)(rg >> 3) * T_TILES + rank * 16 + 2 * sb_) * A_TILE_BYTES + (rg & 7) * 4096 + lane * 16;
            pc.flag = (GAS unsigned*)(dws + WS_FLAGS + (size_t)((rank * 8 + sb_) * 4 + (rg >> 3)) * 128) + (rg & 7);
            pc.src = x + (size_t)(dest * MLOC + rg * 32 + (lane & 31)) * KC + sb_ * 128 + (lane >> 5) * 8;
            return pc;
        };
        f4 cur[2][16];
        auto ldx = [&](int buf, const float* sp) {
#pragma unroll
            for (int j = 0; j < 8; ++j) { cur[buf][2 * j] = *(const f4*)(sp + j * 16); cur[buf][2 * j + 1] = *(const f4*)(sp + j * 16 + 4); }
        };
        auto cvst = [&](int buf, GAS unsigned char* dst, bool remote) {
#pragma unroll
            for (int j = 0; j < 8; ++j) {
                h8 v;
#pragma unroll
                for (int e = 0; e < 4; ++e) { v[e] = (_Float16)cur[buf][2 * j][e]; v[4 + e] = (_Float16)cur[buf][2 * j + 1][e]; }
                asm volatile("global_store_dwordx4 %0, %1, off sc0 sc1\n\ts_nop 1" :: "v"(dst + (size_t)(j >> 2) * A_TILE_BYTES + (j & 3) * 1024), "v"(v) : "memory");
                if (remote && j != 7) asm volatile("s_waitcnt vmcnt(0)" ::: "memory");
            }
        };
        Piece pcs[8];
#pragma unroll
        for (int i = 0; i < 8; ++i) pcs[i] = piece(i);
        ldx(0, pcs[0].src); ldx(1, pcs[1].src);
#pragma unroll
        for (int i = 0; i < 8; ++i) {
            cvst(i & 1, pcs[i].dst, i > 0);
            if (i + 2 < 8) { ldx(i & 1, pcs[i + 2].src); asm volatile("s_waitcnt vmcnt(16)" ::: "memory"); }
            else asm volatile("s_waitcnt vmcnt(0)" ::: "memory");
            if (lane == 0) __hip_atomic_store(pcs[i].flag, call, __ATOMIC_RELAXED, __HIP_MEMORY_SCOPE_SYSTEM);
            if (i == 0) lds_st(CTL + 9, 1u);
        }
    } else {
        GAS unsigned* flags = (GAS unsigned*)(ws + WS_FLAGS);
        int un = 0;
        unsigned long long t0 = rt(); int sp = 0;
        while (un < NUNITS) {
            const int u = un + (lane >> 1);
            bool ok = true;
            if (u < NUNITS) {
                int p, s;
                if (u < 8) { p = rank; s = u; } else { const int v = u - 8; s = v / 7; p = (rank + (v - s * 7) + 1) & 7; }
                typedef unsigned u4v __attribute__((ext_vector_type(4)));
                u4v fv;
                asm volatile("global_load_dwordx4 %0, %1, off sc0 sc1\n\ts_waitcnt vmcnt(0)" : "=v"(fv) : "v"(flags + (size_t)((p * 8 + s) * 4 + mt) * 32 + (lane & 1) * 4) : "memory");
                ok = fv.x >= call && fv.y >= call && fv.z >= call && fv.w >= call;
            }
            const unsigned long long nr = __ballot(!ok);
            const int nready = nr ? (__builtin_ctzll(nr) >> 1) : 32;
            if (nready > 0) {
                un = min(un + nready, NUNITS);
                lds_st(CTL + 0, (unsigned)(2 * un));
                t0 = rt(); sp = 0;
            } else {
                __builtin_amdgcn_s_sleep(2);
                if ((++sp & 63) == 0) {
                    if (lds_ld(CTL + 3) != 0u) break;
                    if (__hip_atomic_load(tmo, __ATOMIC_RELAXED, __HIP_MEMORY_SCOPE_AGENT) == call || (long long)(rt() - t0) > DEADLINE) {
                        __hip_atomic_store(tmo, call, __ATOMIC_RELAXED, __HIP_MEMORY_SCOPE_AGENT);
                        lds_st(CTL + 3, 1u);
                        break;
                    }
                }
            }
        }
    }
}

}

extern "C" void kernel_launch(int rank, int nranks, ncclComm_t comm, hipStream_t stream,
                              void* const* d_in, const int* in_sizes, int n_in,
                              void* d_out, int out_size,
                              const int* in_dtypes, int out_dtype,
                              void* const* d_out_all,
                              void* const* d_ws_all, size_t ws_bytes) {
    (void)comm; (void)in_sizes; (void)n_in; (void)out_size; (void)in_dtypes; (void)out_dtype; (void)d_out_all;
    if (nranks != NRANKS || (unsigned)rank >= (unsigned)NRANKS || ws_bytes < WS_NEED) {
        fprintf(stderr, "kernel_launch: unsupported configuration (nranks %d, rank %d, ws %zu)\n", nranks, rank, ws_bytes);
        return;
    }
    static bool attr_done[64] = {};
    int dev = 0;
    HIP_CHECK(hipGetDevice(&dev));
    if (!attr_done[dev & 63]) {
        HIP_CHECK(hipFuncSetAttribute((const void*)a2a_gemm_relu, hipFuncAttributeMaxDynamicSharedMemorySize, LDS_BYTES));
        attr_done[dev & 63] = true;
    }
    a2a_gemm_relu<<<NWG, 512, LDS_BYTES, stream>>>((const float*)d_in[0], (const float*)d_in[1], (float*)d_out,
                                                   (unsigned char* const*)d_ws_all, rank);
    HIP_CHECK(hipGetLastError());
}
```
